```python
import numpy as np

kernel_name = "gpu_gemm_ar_m4096_k4096_n4096_e4m3e4m3_dequant"


ABSMAX_RTOL = 0.03

COMPUTE_DTYPE = "float32"
EPILOGUE = "dequant"
IN_QUANT = None
OUT_QUANT = None
IN_FP8 = "e4m3e4m3"
IN_DTYPES = {"x": "e4m3", "w_mat": "e4m3",
             "scale_x": "float32", "scale_w": "float32"}
WORLD_SIZE = 8

def _snap_fp8(x, mant_bits, max_abs, min_normal):
    a = np.abs(x.astype(np.float64))
    _m, _e = np.frexp(a)
    q = np.where(a >= min_normal, np.ldexp(1.0, _e - 1 - mant_bits),
                 min_normal * 2.0 ** -mant_bits)
    snapped = np.minimum(np.round(a / q) * q, max_abs)
    return (np.sign(x) * snapped).astype(np.float32)

def setup_inputs():

    x = _snap_fp8(np.random.randn(4096, 4096).astype(np.float32), 3, 448.0, 2.0 ** -6)
    w_mat = _snap_fp8(np.random.randn(4096, 4096).astype(np.float32), 3, 448.0, 2.0 ** -6)
    scale_x = (0.02 * (1.0 + np.random.rand(1))).astype(np.float32)
    scale_w = (0.02 * (1.0 + np.random.rand(1))).astype(np.float32)
    return {"x": x, "w_mat": w_mat, "scale_x": scale_x, "scale_w": scale_w}

def reference(x, w_mat, scale_x, scale_w):
    acc = (x @ w_mat).astype(np.float32)
    y = (acc * (scale_x * scale_w)).astype(np.float32)
    return y
```

```cpp
#include <hip/hip_runtime.h>
#include <rccl/rccl.h>
#include <cstdio>

#define HIP_CHECK(expr) do { hipError_t _e = (expr); if (_e != hipSuccess) { \
    fprintf(stderr, "HIP error %s at %s:%d\n", hipGetErrorString(_e), __FILE__, __LINE__); \
    return; } } while (0)

typedef int v8i __attribute__((ext_vector_type(8)));
typedef int v4i __attribute__((ext_vector_type(4)));
typedef unsigned v4u __attribute__((ext_vector_type(4)));
typedef float v16f __attribute__((ext_vector_type(16)));
typedef float v4f __attribute__((ext_vector_type(4)));
typedef unsigned long long u64;
#define LAS __attribute__((address_space(3)))
#define GAS __attribute__((address_space(1)))

namespace {
constexpr int NR = 8;
constexpr int MN = 4096;
constexpr int KS = 512;
constexpr int NTHREADS = 768;
constexpr int NWG = 256;
constexpr unsigned PANEL = 16384;
constexpr unsigned OPB = 16u << 20;
constexpr unsigned GB = 2 * OPB;
constexpr size_t OFF_CTR = 0;
constexpr size_t OFF_TMO = 32768;
constexpr size_t OFF_SF = 65536;
constexpr size_t OFF_RF = 131072;
constexpr size_t OFF_DBG = 262144;
constexpr size_t OFF_G = 1u << 20;
constexpr size_t WS_NEED = OFF_G + 2 * (size_t)GB;
constexpr int NSLOT = 4;
constexpr unsigned SLOTB = 2 * PANEL;
constexpr unsigned L_CTL = NSLOT * SLOTB;
constexpr unsigned L_FULL = L_CTL;
constexpr unsigned L_FREE = L_CTL + 64;
constexpr unsigned L_ORDER = L_CTL + 128;
constexpr unsigned L_CALL = L_CTL + 512;
constexpr unsigned L_ABORT = L_CTL + 516;
constexpr unsigned LDS_BYTES = L_CTL + 1024;
constexpr u64 DEADLINE = 50000000ull;
#define NWL 18

struct Params {
    const unsigned char* x;
    const unsigned char* w;
    const float* sx;
    const float* sw;
    float* out;
    char* ws[NR];
    int rank;
};

__device__ __forceinline__ __amdgpu_buffer_rsrc_t mkrsrc(const void* p, unsigned bytes) {
    return __builtin_amdgcn_make_buffer_rsrc((void*)p, 0, bytes, 0x00020000);
}
__device__ __forceinline__ u64 rt() { return __builtin_amdgcn_s_memrealtime(); }
__device__ __forceinline__ unsigned rfl(unsigned v) { return __builtin_amdgcn_readfirstlane(v); }
__device__ __forceinline__ unsigned ld_agent(const char* p) {
    return __hip_atomic_load((const GAS unsigned*)p, __ATOMIC_RELAXED, __HIP_MEMORY_SCOPE_AGENT);
}
__device__ __forceinline__ unsigned ld_sys(const char* p) {
    return __hip_atomic_load((const GAS unsigned*)p, __ATOMIC_RELAXED, __HIP_MEMORY_SCOPE_SYSTEM);
}
__device__ __forceinline__ void st_agent(char* p, unsigned v) {
    __hip_atomic_store((GAS unsigned*)p, v, __ATOMIC_RELAXED, __HIP_MEMORY_SCOPE_AGENT);
}
__device__ __forceinline__ void st_sys(char* p, unsigned v) {
    __hip_atomic_store((GAS unsigned*)p, v, __ATOMIC_RELAXED, __HIP_MEMORY_SCOPE_SYSTEM);
}
#define WAITV(n) asm volatile("s_waitcnt vmcnt(" #n ")" ::: "memory")
#define WAITL0() asm volatile("s_waitcnt lgkmcnt(0)" ::: "memory")

#define STAMP(idx) do { } while (0)

__device__ __forceinline__ void dma4k(__amdgpu_buffer_rsrc_t r, unsigned voff, unsigned soff, unsigned lds) {
    asm volatile(
        "s_mov_b32 m0, %0\n\t"
        "s_nop 3\n\t"
        "buffer_load_dwordx4 %1, %2, %3 offen sc1 lds\n\t"
        "buffer_load_dwordx4 %1, %2, %3 offen offset:1024 sc1 lds\n\t"
        "buffer_load_dwordx4 %1, %2, %3 offen offset:2048 sc1 lds\n\t"
        "buffer_load_dwordx4 %1, %2, %3 offen offset:3072 sc1 lds"
        :: "s"(lds), "v"(voff), "s"(r), "s"(soff) : "memory");
}

__device__ __forceinline__ v8i ldfrag(const LAS char* p, int o0, int o1) {
    v4i a = *(const LAS v4i*)(p + o0);
    v4i b = *(const LAS v4i*)(p + o1);
    return __builtin_shufflevector(a, b, 0, 1, 2, 3, 4, 5, 6, 7);
}
#define MFMA(a, b, c) __builtin_amdgcn_mfma_scale_f32_32x32x64_f8f6f4((a), (b), (c), 0, 0, 0, 0, 0, 0)

__device__ __forceinline__ unsigned panel_of(unsigned n) {
    unsigned t = n >> 5, blk = (n >> 1) & 15, op = (n & 1) ^ 1;
    return (op << 7) | (t << 4) | blk;
}
}

__global__ void __launch_bounds__(NTHREADS, 3) gemm_ag_kernel(Params P) {
    __shared__ __attribute__((aligned(16))) char smem[LDS_BYTES];
    LAS char* L = (LAS char*)smem;
    const int tid = threadIdx.x;
    const int wave = rfl(tid >> 6);
    const int lane = tid & 63;
    const int b = blockIdx.x;
    const int rank = P.rank;
    char* myws = P.ws[rank];

    const int xcd = b & 7, idx = b >> 3;
    const int ti = (xcd >> 1) + 4 * (idx >> 3);
    const int tj = (xcd & 1) * 8 + (idx & 7);

    if (wave >= 8) {
        const int e = wave - 8;
        STAMP(e);
        __builtin_amdgcn_s_setprio(2);
        const unsigned sp = b, sop = sp >> 7, st = (sp >> 4) & 7, sblk = sp & 15;
        v4u o[4];
        if (sop == 0) {
            auto rx = mkrsrc(P.x, MN * KS);
            unsigned base = (sblk * 256 + e * 64 + (lane & 31)) * KS + st * 64 + (lane >> 5) * 16;
            o[0] = __builtin_amdgcn_raw_buffer_load_b128(rx, base, 0, 0);
            o[1] = __builtin_amdgcn_raw_buffer_load_b128(rx, base + 32, 0, 0);
            o[2] = __builtin_amdgcn_raw_buffer_load_b128(rx, base + 32 * KS, 0, 0);
            o[3] = __builtin_amdgcn_raw_buffer_load_b128(rx, base + 32 * KS + 32, 0, 0);
        } else {
            auto rw = mkrsrc(P.w, MN * KS);
            unsigned base = (st * 64 + (e & 1) * 32 + (lane >> 5) * 16) * MN + sblk * 256 + (e >> 1) * 128 + (lane & 31) * 4;
            unsigned d[16];
#pragma unroll
            for (int i = 0; i < 16; ++i) d[i] = __builtin_amdgcn_raw_buffer_load_b32(rw, base, i * MN, 0);
#pragma unroll
            for (int g = 0; g < 4; ++g) {
                unsigned t0 = __builtin_amdgcn_perm(d[4 * g + 1], d[4 * g], 0x05010400u);
                unsigned t1 = __builtin_amdgcn_perm(d[4 * g + 1], d[4 * g], 0x07030602u);
                unsigned t2 = __builtin_amdgcn_perm(d[4 * g + 3], d[4 * g + 2], 0x05010400u);
                unsigned t3 = __builtin_amdgcn_perm(d[4 * g + 3], d[4 * g + 2], 0x07030602u);
                o[0][g] = __builtin_amdgcn_perm(t2, t0, 0x05040100u);
                o[1][g] = __builtin_amdgcn_perm(t2, t0, 0x07060302u);
                o[2][g] = __builtin_amdgcn_perm(t3, t1, 0x05040100u);
                o[3][g] = __builtin_amdgcn_perm(t3, t1, 0x07060302u);
            }
        }
        if (e == 0) {
            unsigned c = 0;
            if (lane == 0) {
                c = ld_agent(myws + OFF_CTR + b * 128) + 1;
                st_agent(myws + OFF_CTR + b * 128, c);
                *(volatile LAS unsigned*)(L + L_CALL) = c;
                *(volatile LAS unsigned*)(L + L_ABORT) = 0;
            }
        }
        WAITL0();
        __builtin_amdgcn_s_barrier();
        const unsigned call = rfl(*(volatile LAS unsigned*)(L + L_CALL));
        const unsigned par = call & 1;
        auto rg = mkrsrc(myws + OFF_G + (size_t)par * GB, GB);
        {
            unsigned off = sop * OPB + ((rank * 8 + st) * 16 + sblk) * PANEL + e * 4096 + lane * 16;
#pragma unroll
            for (int i = 0; i < 4; ++i) __builtin_amdgcn_raw_buffer_store_b128(o[i], rg, off + i * 1024, 0, 17);
            WAITV(0);
            if (lane == 0) {
#pragma unroll
                for (int d = 1; d < NR; ++d) {
                    int p = (rank + d) & 7;
                    st_sys(P.ws[p] + OFF_SF + ((rank * 256 + sp) * 4 + e) * 4, call);
                }
                st_agent(myws + OFF_RF + (((sop * 16 + sblk) * 64 + rank * 8 + st) * 4 + e) * 4, call);
            }
        }
        STAMP(4 + e);
        const u64 t0 = rt();

        if (e == 0) {
            auto rf = mkrsrc(myws + OFF_RF, 32768);
            const unsigned ox = (ti * 64 + lane) * 16, ow = ((16 + tj) * 64 + lane) * 16;
            u64 seen = 0;
            unsigned cnt = 0, idle = 0;
            while (seen != ~0ull) {
                bool ok = false;
                if (!((seen >> lane) & 1)) {
                    v4u a = __builtin_amdgcn_raw_buffer_load_b128(rf, ox, 0, 16);
                    v4u c = __builtin_amdgcn_raw_buffer_load_b128(rf, ow, 0, 16);
                    unsigned m = min(min(min(a.x, a.y), min(a.z, a.w)), min(min(c.x, c.y), min(c.z, c.w)));
                    ok = m >= call;
                }
                u64 nw = __ballot(ok);
                if (nw == 0) {
                    if ((++idle & 15) == 0) {
                        if (rfl(ld_agent(myws + OFF_TMO)) == call || rt() - t0 > DEADLINE) {
                            if (lane == 0) { st_agent(myws + OFF_TMO, call); *(volatile LAS unsigned*)(L + L_ABORT) = 1; }
                            nw = ~seen;
                            ok = !((seen >> lane) & 1);
                        }
                    }
                    if (nw == 0) { __builtin_amdgcn_s_sleep(4); continue; }
                }
                unsigned pos = cnt + __builtin_amdgcn_mbcnt_hi((unsigned)(nw >> 32), __builtin_amdgcn_mbcnt_lo((unsigned)nw, 0));
                if (ok) *(volatile LAS unsigned*)(L + L_ORDER + pos * 4) = lane + 1;
                cnt += __builtin_popcountll(nw);
                seen |= nw;
            }
            STAMP(8);
        } else if (e <= 2) {
            const unsigned op = e - 1;
            const unsigned blk = op ? tj : ti;
            const unsigned voff = lane * 16;
            unsigned issued = 0, pub = 0, spins = 0;
            while (pub < 64) {
                bool did = false;
                if (issued < 64 && issued - pub < 3) {
                    unsigned addr = lane < 8 ? L_FREE + lane * 4 : L_ORDER + issued * 4;
                    unsigned v = *(volatile LAS unsigned*)(L + addr);
                    unsigned q1 = __builtin_amdgcn_readlane(v, 8);
                    unsigned need = issued >= NSLOT ? issued - (NSLOT - 1) : 0;
                    bool fr = (__ballot(v >= need) & 0xff) == 0xff;
                    if (q1 != 0 && fr) {
                        unsigned soff = op * OPB + ((q1 - 1) * 16 + blk) * PANEL;
                        unsigned lds = (unsigned)(size_t)L + (issued & (NSLOT - 1)) * SLOTB + op * PANEL;
                        dma4k(rg, voff, soff, lds);
                        dma4k(rg, voff, soff + 4096, lds + 4096);
                        dma4k(rg, voff, soff + 8192, lds + 8192);
                        dma4k(rg, voff, soff + 12288, lds + 12288);
                        ++issued;
                        did = true;
                    }
                }
                if (!did) {
                    if (pub < issued) {
                        unsigned infl = issued - pub;
                        if (infl == 1) WAITV(0);
                        else if (infl == 2) WAITV(16);
                        else WAITV(32);
                        ++pub;
                        *(volatile LAS unsigned short*)(L + L_FULL + op * 2) = (unsigned short)pub;
                    } else {
                        __builtin_amdgcn_s_sleep(1);
                        if ((++spins & 1023) == 0 && rt() - t0 > 2 * DEADLINE) break;
                    }
                }
            }
            STAMP(8 + e);
        } else {
            if (b < 7 * NWL) {
                const unsigned link = b % 7, widx = b / 7;
                const unsigned src = (rank + 1 + link) & 7;
                auto rs = mkrsrc(P.ws[src] + OFF_G + (size_t)par * GB, GB);
                const char* sf = myws + OFF_SF + src * 4096;
                const unsigned voff = lane * 16;
                auto unit_off = [&](unsigned n) {
                    unsigned p = panel_of(n >> 2);
                    return (p >> 7) * OPB + ((src * 8 + ((p >> 4) & 7)) * 16 + (p & 15)) * PANEL + (n & 3) * 4096;
                };
                auto sf_addr = [&](unsigned n) { return sf + (panel_of(n >> 2) * 4 + (n & 3)) * 4; };
                auto rf_addr = [&](unsigned n) {
                    unsigned p = panel_of(n >> 2);
                    return myws + OFF_RF + ((((p >> 7) * 16 + (p & 15)) * 64 + src * 8 + ((p >> 4) & 7)) * 4 + (n & 3)) * 4;
                };
                auto wait_sf = [&](unsigned n) -> bool {
                    unsigned sp_ = 0;
                    while (rfl(ld_sys(sf_addr(n))) < call) {
                        if ((++sp_ & 15) == 0 && (rfl(ld_agent(myws + OFF_TMO)) == call || rt() - t0 > DEADLINE)) {
                            if (lane == 0) st_agent(myws + OFF_TMO, call);
                            return false;
                        }
                        __builtin_amdgcn_s_sleep(1);
                    }
                    return true;
                };
                unsigned n = widx;
                bool alive = wait_sf(n);
                STAMP(12);
                if (alive) {
                    v4u v[4];
                    unsigned so = unit_off(n);
#pragma unroll
                    for (int i = 0; i < 4; ++i) v[i] = __builtin_amdgcn_raw_buffer_load_b128(rs, so + voff + i * 1024, 0, 17);
                    unsigned n2 = n + NWL;
                    unsigned f = ld_sys(sf_addr(n2 < 1024 ? n2 : n));
                    while (true) {
#pragma unroll
                        for (int i = 0; i < 4; ++i) __builtin_amdgcn_raw_buffer_store_b128(v[i], rg, so + voff + i * 1024, 0, 16);
                        if (n2 >= 1024) {
                            WAITV(0);
                            if (lane == 0) st_agent(rf_addr(n), call);
                            break;
                        }
                        bool flagged = false;
                        if (rfl(f) < call) {
                            WAITV(0);
                            if (lane == 0) st_agent(rf_addr(n), call);
                            flagged = true;
                            if (!wait_sf(n2)) break;
                        }
                        so = unit_off(n2);
#pragma unroll
                        for (int i = 0; i < 4; ++i) v[i] = __builtin_amdgcn_raw_buffer_load_b128(rs, so + voff + i * 1024, 0, 17);
                        unsigned n3 = n2 + NWL;
                        f = ld_sys(sf_addr(n3 < 1024 ? n3 : n2));
                        if (!flagged) {
                            WAITV(5);
                            if (lane == 0) st_agent(rf_addr(n), call);
                        }
                        n = n2;
                        n2 = n3;
                    }
                }
                STAMP(13);
            }
        }
    } else {
        if (tid < 128) *(volatile LAS unsigned*)(L + L_CTL + tid * 4) = 0;
        const float scale = P.sx[0] * P.sw[0];
        WAITL0();
        __builtin_amdgcn_s_barrier();
        const int wm = wave >> 1, wn = wave & 1;
        v16f acc[2][4];
#pragma unroll
        for (int i = 0; i < 2; ++i)
#pragma unroll
            for (int j = 0; j < 4; ++j)
#pragma unroll
                for (int r = 0; r < 16; ++r) acc[i][j][r] = 0.f;
        const LAS char* xa = L + wm * 4096 + lane * 16;
        const LAS char* wb = L + PANEL + wn * 8192 + lane * 16;
        volatile LAS unsigned* fullp = (volatile LAS unsigned*)(L + L_FULL);
        volatile LAS unsigned* freep = (volatile LAS unsigned*)(L + L_FREE + wave * 4);
        unsigned fw = 0;
        for (unsigned pos = 0; pos < 64; ++pos) {
            unsigned f = rfl(fw);
            unsigned avail = min(f & 0xffffu, f >> 16);
            if (avail <= pos) {
                unsigned spins = 0;
                const u64 t0 = rt();
                while (true) {
                    f = rfl(*fullp);
                    avail = min(f & 0xffffu, f >> 16);
                    if (avail > pos) break;
                    __builtin_amdgcn_s_sleep(1);
                    if ((++spins & 1023) == 0 && rt() - t0 > 3 * DEADLINE) break;
                }
            }
            asm volatile("" ::: "memory");
            const unsigned so = (pos & (NSLOT - 1)) * SLOTB;
            const LAS char* xs = xa + so;
            const LAS char* wsl = wb + so;
            v8i x0 = ldfrag(xs, 0, 1024);
            v8i w0 = ldfrag(wsl, 0, 4096);
            v8i x1 = ldfrag(xs, 2048, 3072);
            v8i w1 = ldfrag(wsl, 1024, 5120);
            acc[0][0] = MFMA(x0, w0, acc[0][0]);
            acc[1][0] = MFMA(x1, w0, acc[1][0]);
            v8i w2 = ldfrag(wsl, 2048, 6144);
            acc[0][1] = MFMA(x0, w1, acc[0][1]);
            acc[1][1] = MFMA(x1, w1, acc[1][1]);
            v8i w3 = ldfrag(wsl, 3072, 7168);
            acc[0][2] = MFMA(x0, w2, acc[0][2]);
            acc[1][2] = MFMA(x1, w2, acc[1][2]);
            asm volatile("s_waitcnt lgkmcnt(0)" : "+v"(w3) :: "memory");
            *freep = pos + 1;
            fw = *fullp;
            asm volatile("" : "+v"(w3) :: "memory");
            acc[0][3] = MFMA(x0, w3, acc[0][3]);
            acc[1][3] = MFMA(x1, w3, acc[1][3]);
        }
        if (wave == 0) STAMP(16);
        const bool bad = rfl(*(volatile LAS unsigned*)(L + L_ABORT)) != 0;
        const float s = bad ? __builtin_nanf("") : scale;
        auto ro = mkrsrc((char*)P.out + ((size_t)(ti * 256 + wm * 64) * MN + tj * 256 + wn * 128) * 4, 64u * MN * 4u);
        const unsigned vo = (lane >> 5) * 4 * (MN * 4) + (lane & 31) * 16;
#pragma unroll
        for (int mb = 0; mb < 2; ++mb)
#pragma unroll
            for (int r = 0; r < 16; ++r) {
                v4f ov = {acc[mb][0][r] * s, acc[mb][1][r] * s, acc[mb][2][r] * s, acc[mb][3][r] * s};
                __builtin_amdgcn_raw_buffer_store_b128(__builtin_bit_cast(v4u, ov), ro,
                                                       vo + (mb * 32 + (r & 3) + 8 * (r >> 2)) * (MN * 4), 0, 16);
            }
    }
}

extern "C" void kernel_launch(int rank, int nranks, ncclComm_t comm,
                              hipStream_t stream,
                              void* const* d_in, const int* in_sizes, int n_in,
                              void* d_out, int out_size,
                              const int* in_dtypes, int out_dtype,
                              void* const* d_out_all,
                              void* const* d_ws_all, size_t ws_bytes) {
    if (nranks != NR || (unsigned)rank >= (unsigned)NR || n_in != 4 || ws_bytes < WS_NEED ||
        in_sizes[0] != MN * KS || in_sizes[1] != MN * KS || out_size != MN * MN) {
        fprintf(stderr, "kernel_launch: unexpected problem shape\n");
        return;
    }
    Params P;
    P.x = (const unsigned char*)d_in[0];
    P.w = (const unsigned char*)d_in[1];
    P.sx = (const float*)d_in[2];
    P.sw = (const float*)d_in[3];
    P.out = (float*)d_out;
    for (int p = 0; p < NR; ++p) P.ws[p] = (char*)d_ws_all[p];
    P.rank = rank;
    hipLaunchKernelGGL(gemm_ag_kernel, dim3(NWG), dim3(NTHREADS), 0, stream, P);
    HIP_CHECK(hipGetLastError());
}
```

```cpp
#include <hip/hip_runtime.h>
#include <rccl/rccl.h>
#include <cstdio>

#define HIP_CHECK(expr) do { hipError_t _e = (expr); if (_e != hipSuccess) { \
    fprintf(stderr, "HIP error %s at %s:%d\n", hipGetErrorString(_e), __FILE__, __LINE__); \
    return; } } while (0)

typedef int v8i __attribute__((ext_vector_type(8)));
typedef int v4i __attribute__((ext_vector_type(4)));
typedef unsigned v4u __attribute__((ext_vector_type(4)));
typedef float v16f __attribute__((ext_vector_type(16)));
typedef float v4f __attribute__((ext_vector_type(4)));
typedef unsigned long long u64;
#define LAS __attribute__((address_space(3)))
#define GAS __attribute__((address_space(1)))

namespace {
constexpr int NR = 8;
constexpr int MN = 4096;
constexpr int KS = 512;
constexpr int NTHREADS = 768;
constexpr int NWG = 256;
constexpr unsigned UNIT = 4096;
constexpr unsigned QB = 64 * UNIT;
constexpr unsigned OPB = 64 * QB;
constexpr unsigned GB = 2 * OPB;
constexpr size_t OFF_CTR = 0;
constexpr size_t OFF_TMO = 32768;
constexpr size_t OFF_SF = 65536;
constexpr size_t OFF_RFX = 131072;
constexpr size_t OFF_RFW = 131072 + 32768;
constexpr size_t OFF_DBG = 262144;
constexpr size_t OFF_G = 1u << 20;
constexpr size_t WS_NEED = OFF_G + 2 * (size_t)GB;
constexpr int NSLOT = 4;
constexpr unsigned XB = 5 * UNIT;
constexpr unsigned SLOTB = 9 * UNIT;
constexpr unsigned L_CTL = NSLOT * SLOTB;
constexpr unsigned L_FULL = L_CTL;
constexpr unsigned L_FREE = L_CTL + 64;
constexpr unsigned L_ORDER = L_CTL + 128;
constexpr unsigned L_CALL = L_CTL + 512;
constexpr unsigned L_ABORT = L_CTL + 516;
constexpr unsigned LDS_BYTES = L_CTL + 1024;
constexpr u64 DEADLINE = 50000000ull;
#define DW 3
#define DC 6
#define ORD 0

struct Params {
    const unsigned char* x;
    const unsigned char* w;
    const float* sx;
    const float* sw;
    float* out;
    char* ws[NR];
    int rank;
};

__device__ __forceinline__ __amdgpu_buffer_rsrc_t mkrsrc(const void* p, unsigned bytes) {
    return __builtin_amdgcn_make_buffer_rsrc((void*)p, 0, bytes, 0x00020000);
}
__device__ __forceinline__ u64 rt() { return __builtin_amdgcn_s_memrealtime(); }
__device__ __forceinline__ unsigned rfl(unsigned v) { return __builtin_amdgcn_readfirstlane(v); }
__device__ __forceinline__ unsigned ld_agent(const char* p) {
    return __hip_atomic_load((const GAS unsigned*)p, __ATOMIC_RELAXED, __HIP_MEMORY_SCOPE_AGENT);
}
__device__ __forceinline__ unsigned ld_sys(const char* p) {
    return __hip_atomic_load((const GAS unsigned*)p, __ATOMIC_RELAXED, __HIP_MEMORY_SCOPE_SYSTEM);
}
__device__ __forceinline__ void st_agent(char* p, unsigned v) {
    __hip_atomic_store((GAS unsigned*)p, v, __ATOMIC_RELAXED, __HIP_MEMORY_SCOPE_AGENT);
}
__device__ __forceinline__ void st_sys(char* p, unsigned v) {
    __hip_atomic_store((GAS unsigned*)p, v, __ATOMIC_RELAXED, __HIP_MEMORY_SCOPE_SYSTEM);
}
#define WAITV(n) asm volatile("s_waitcnt vmcnt(" #n ")" ::: "memory")
#define WAITL0() asm volatile("s_waitcnt lgkmcnt(0)" ::: "memory")

#define STAMP(idx) do { } while (0)

__device__ __forceinline__ void dma4k(__amdgpu_buffer_rsrc_t r, unsigned voff, unsigned soff, unsigned lds) {
    asm volatile(
        "s_mov_b32 m0, %0\n\t"
        "s_nop 3\n\t"
        "buffer_load_dwordx4 %1, %2, %3 offen sc1 lds\n\t"
        "buffer_load_dwordx4 %1, %2, %3 offen offset:1024 sc1 lds\n\t"
        "buffer_load_dwordx4 %1, %2, %3 offen offset:2048 sc1 lds\n\t"
        "buffer_load_dwordx4 %1, %2, %3 offen offset:3072 sc1 lds"
        :: "s"(lds), "v"(voff), "s"(r), "s"(soff) : "memory");
}

__device__ __forceinline__ v8i ldfrag(const LAS char* p, int o0, int o1) {
    v4i a = *(const LAS v4i*)(p + o0);
    v4i b = *(const LAS v4i*)(p + o1);
    return __builtin_shufflevector(a, b, 0, 1, 2, 3, 4, 5, 6, 7);
}
#define MFMA(a, b, c) __builtin_amdgcn_mfma_scale_f32_32x32x64_f8f6f4((a), (b), (c), 0, 0, 0, 0, 0, 0)

__device__ __forceinline__ unsigned unit_of(unsigned n, unsigned link) {
    unsigned t = ((n >> 7) + link) & 7, blk = ((n >> 3) + (link * 16) / 7) & 15, op = ((n >> 2) & 1) ^ 1;
    return (op << 9) | (t << 6) | (blk << 2) | (n & 3);
}
__device__ __forceinline__ unsigned rf_off(unsigned op, unsigned q, unsigned u) {
    unsigned ti = (u * 205u) >> 10;
    return op ? (unsigned)OFF_RFW + (((u >> 2) * 64 + q) * 4 + (u & 3)) * 4
              : (unsigned)OFF_RFX + ((ti * 64 + q) * 8 + (u - ti * 5)) * 4;
}
}

__global__ void __launch_bounds__(NTHREADS, 3) gemm_ag_kernel(Params P) {
    __shared__ __attribute__((aligned(16))) char smem[LDS_BYTES];
    LAS char* L = (LAS char*)smem;
    const int tid = threadIdx.x;
    const int wave = rfl(tid >> 6);
    const int lane = tid & 63;
    const int b = blockIdx.x;
    const int rank = P.rank;
    char* myws = P.ws[rank];

    {
        const int e = wave - 8;
        const unsigned sop = b >> 7, st = (b >> 4) & 7, sblk = b & 15;
        v4u o[4];
        if (e >= 0) {
            STAMP(e);
            if (sop == 0) {
                auto rx = mkrsrc(P.x, MN * KS);
                unsigned base = (sblk * 256 + e * 64 + (lane & 31)) * KS + st * 64 + (lane >> 5) * 16;
                o[0] = __builtin_amdgcn_raw_buffer_load_b128(rx, base, 0, 0);
                o[1] = __builtin_amdgcn_raw_buffer_load_b128(rx, base + 32, 0, 0);
                o[2] = __builtin_amdgcn_raw_buffer_load_b128(rx, base + 32 * KS, 0, 0);
                o[3] = __builtin_amdgcn_raw_buffer_load_b128(rx, base + 32 * KS + 32, 0, 0);
            } else {
                auto rw = mkrsrc(P.w, MN * KS);
                unsigned base = (st * 64 + (e & 1) * 32 + (lane >> 5) * 16) * MN + sblk * 256 + (e >> 1) * 128 + (lane & 31) * 4;
                unsigned d[16];
#pragma unroll
                for (int i = 0; i < 16; ++i) d[i] = __builtin_amdgcn_raw_buffer_load_b32(rw, base, i * MN, 0);
#pragma unroll
                for (int g = 0; g < 4; ++g) {
                    unsigned t0 = __builtin_amdgcn_perm(d[4 * g + 1], d[4 * g], 0x05010400u);
                    unsigned t1 = __builtin_amdgcn_perm(d[4 * g + 1], d[4 * g], 0x07030602u);
                    unsigned t2 = __builtin_amdgcn_perm(d[4 * g + 3], d[4 * g + 2], 0x05010400u);
                    unsigned t3 = __builtin_amdgcn_perm(d[4 * g + 3], d[4 * g + 2], 0x07030602u);
                    o[0][g] = __builtin_amdgcn_perm(t2, t0, 0x05040100u);
                    o[1][g] = __builtin_amdgcn_perm(t2, t0, 0x07060302u);
                    o[2][g] = __builtin_amdgcn_perm(t3, t1, 0x05040100u);
                    o[3][g] = __builtin_amdgcn_perm(t3, t1, 0x07060302u);
                }
            }
        } else if (wave == 0) {
            if (lane == 0) {
                unsigned c = ld_agent(myws + OFF_CTR + b * 128) + 1;
                st_agent(myws + OFF_CTR + b * 128, c);
                *(volatile LAS unsigned*)(L + L_CALL) = c;
                *(volatile LAS unsigned*)(L + L_ABORT) = 0;
            }
        } else if (wave <= 2) {
            *(volatile LAS unsigned*)(L + L_CTL + (tid - 64) * 4) = 0;
        }
        WAITL0();
        __builtin_amdgcn_s_barrier();
        if (e >= 0) {
            const unsigned call = rfl(*(volatile LAS unsigned*)(L + L_CALL));
            auto rg = mkrsrc(myws + OFF_G + (size_t)(call & 1) * GB, GB);
            const unsigned q = rank * 8 + st, u = sblk * 4 + e;
            unsigned off = sop * OPB + q * QB + u * UNIT + lane * 16;
#pragma unroll
            for (int i = 0; i < 4; ++i) __builtin_amdgcn_raw_buffer_store_b128(o[i], rg, off + i * 1024, 0, 17);
            WAITV(0);
            if (lane == 0) {
#pragma unroll
                for (int d = 1; d < NR; ++d) {
                    int p = (rank + d) & 7;
                    st_sys(P.ws[p] + OFF_SF + (rank * 1024 + b * 4 + e) * 4, call);
                }
                st_agent(myws + rf_off(sop, q, u), call);
            }
            STAMP(4 + e);
        }
    }
    const unsigned call = rfl(*(volatile LAS unsigned*)(L + L_CALL));
    const unsigned par = call & 1;
    const u64 t0 = rt();
    const int xcd = b & 7, idx = b >> 3;

    if (idx < 6) {
        if (b >= 7 * DC || wave >= DW) return;
        constexpr unsigned NWL = DW * DC;
        auto rg = mkrsrc(myws + OFF_G + (size_t)par * GB, GB);
        const unsigned link = b % 7, widx = (b / 7) * DW + wave;
        const unsigned src = (rank + 1 + link) & 7;
        auto rs = mkrsrc(P.ws[src] + OFF_G + (size_t)par * GB, GB);
        const char* sf = myws + OFF_SF + src * 4096;
        const unsigned voff = lane * 16;
        auto unit_off = [&](unsigned id) { return (id >> 9) * OPB + (src * 8 + ((id >> 6) & 7)) * QB + (id & 63) * UNIT; };
        auto rf_addr = [&](unsigned id) { return myws + rf_off(id >> 9, src * 8 + ((id >> 6) & 7), id & 63); };
        auto wait_sf = [&](unsigned id) -> bool {
            unsigned sp_ = 0;
            while (rfl(ld_sys(sf + id * 4)) < call) {
                if ((++sp_ & 15) == 0 && (rfl(ld_agent(myws + OFF_TMO)) == call || rt() - t0 > DEADLINE)) {
                    if (lane == 0) st_agent(myws + OFF_TMO, call);
                    return false;
                }
                __builtin_amdgcn_s_sleep(1);
            }
            return true;
        };
        unsigned id = unit_of(widx, link);
        bool alive = wait_sf(id);
        STAMP(24 + wave);
        if (alive) {
            v4u v[4];
            unsigned so = unit_off(id);
#pragma unroll
            for (int i = 0; i < 4; ++i) v[i] = __builtin_amdgcn_raw_buffer_load_b128(rs, so + voff + i * 1024, 0, 17);
            unsigned n2 = widx + NWL;
            unsigned id2 = unit_of(n2, link);
            unsigned f = ld_sys(sf + id2 * 4);
            while (true) {
#pragma unroll
                for (int i = 0; i < 4; ++i) __builtin_amdgcn_raw_buffer_store_b128(v[i], rg, so + voff + i * 1024, 0, 16);
                if (n2 >= 1024) {
                    WAITV(0);
                    if (lane == 0) st_agent(rf_addr(id), call);
                    break;
                }
                bool flagged = false;
                if (rfl(f) < call) {
                    WAITV(0);
                    if (lane == 0) st_agent(rf_addr(id), call);
                    flagged = true;
                    if (!wait_sf(id2)) break;
                }
                so = unit_off(id2);
#pragma unroll
                for (int i = 0; i < 4; ++i) v[i] = __builtin_amdgcn_raw_buffer_load_b128(rs, so + voff + i * 1024, 0, 17);
                unsigned n3 = n2 + NWL;
                unsigned id3 = unit_of(n3 & 1023, link);
                f = ld_sys(sf + id3 * 4);
                if (!flagged) {
                    WAITV(5);
                    if (lane == 0) st_agent(rf_addr(id), call);
                }
                id = id2;
                id2 = id3;
                n2 = n3;
            }
        }
        STAMP(40 + wave);
        return;
    }

    const int ti = (idx - 6) >> 1;
    const int tj = xcd * 2 + (idx & 1);
    const bool shortrow = ti == 12;

    if (wave == 10) {
        __builtin_amdgcn_s_setprio(2);
        auto rf = mkrsrc(myws, OFF_DBG);
        const unsigned ox = OFF_RFX + (ti * 64 + lane) * 32, ow = OFF_RFW + (tj * 64 + lane) * 16;
        const unsigned ox4 = shortrow ? ox : ox + 16;
        u64 seen = 0;
        unsigned cnt = 0, idle = 0;
        while (seen != ~0ull) {
            bool ok = false;
            if (!((seen >> lane) & 1)) {
                v4u a = __builtin_amdgcn_raw_buffer_load_b128(rf, ox, 0, 16);
                unsigned a4 = __builtin_amdgcn_raw_buffer_load_b32(rf, ox4, 0, 16);
                v4u c = __builtin_amdgcn_raw_buffer_load_b128(rf, ow, 0, 16);
                unsigned m = min(min(min(a.x, a.y), min(a.z, a.w)), min(min(c.x, c.y), min(c.z, c.w)));
                ok = min(m, a4) >= call;
            }
            u64 nw = __ballot(ok);
            if (nw == 0) {
                if ((++idle & 15) == 0) {
                    if (rfl(ld_agent(myws + OFF_TMO)) == call || rt() - t0 > DEADLINE) {
                        if (lane == 0) { st_agent(myws + OFF_TMO, call); *(volatile LAS unsigned*)(L + L_ABORT) = 1; }
                        nw = ~seen;
                        ok = !((seen >> lane) & 1);
                    }
                }
                if (nw == 0) { __builtin_amdgcn_s_sleep(4); continue; }
            }
            unsigned pos = cnt + __builtin_amdgcn_mbcnt_hi((unsigned)(nw >> 32), __builtin_amdgcn_mbcnt_lo((unsigned)nw, 0));
            if (ok) *(volatile LAS unsigned*)(L + L_ORDER + pos * 4) = lane + 1;
            cnt += __builtin_popcountll(nw);
            seen |= nw;
        }
        STAMP(8);
    } else if (wave == 11) {
        __builtin_amdgcn_s_setprio(2);
        auto rg = mkrsrc(myws + OFF_G + (size_t)par * GB, GB);
        const unsigned voff = lane * 16;
        const unsigned nfree = shortrow ? 0xffu : 0x3ffu;
        unsigned issued = 0, pub = 0, spins = 0;
        while (pub < 64) {
            bool did = false;
            if (issued < 64) {
                unsigned addr = lane < 10 ? L_FREE + lane * 4 : L_ORDER + issued * 4;
                unsigned v = *(volatile LAS unsigned*)(L + addr);
                unsigned q1 = __builtin_amdgcn_readlane(v, 10);
                unsigned need = issued >= NSLOT ? issued - (NSLOT - 1) : 0;
                bool fr = ((unsigned)__ballot(v >= need) & nfree) == nfree;
                if (q1 != 0 && fr) {
                    unsigned sx = (q1 - 1) * QB + ti * XB;
                    unsigned sw = OPB + (q1 - 1) * QB + tj * 4 * UNIT;
                    unsigned lds = (unsigned)(size_t)L + (issued & (NSLOT - 1)) * SLOTB;
                    dma4k(rg, voff, sw, lds + XB);
                    dma4k(rg, voff, sx, lds);
                    dma4k(rg, voff, sw + UNIT, lds + XB + UNIT);
                    dma4k(rg, voff, sx + UNIT, lds + UNIT);
                    dma4k(rg, voff, sw + 2 * UNIT, lds + XB + 2 * UNIT);
                    dma4k(rg, voff, sx + 2 * UNIT, lds + 2 * UNIT);
                    if (pub < issued) {
                        WAITV(24);
                        ++pub;
                        *(volatile LAS unsigned*)(L + L_FULL) = pub;
                    }
                    dma4k(rg, voff, sw + 3 * UNIT, lds + XB + 3 * UNIT);
                    dma4k(rg, voff, sx + 3 * UNIT, lds + 3 * UNIT);
                    if (!shortrow) dma4k(rg, voff, sx + 4 * UNIT, lds + 4 * UNIT);
                    ++issued;
                    did = true;
                }
            }
            if (!did) {
                if (pub < issued) {
                    WAITV(0);
                    ++pub;
                    *(volatile LAS unsigned*)(L + L_FULL) = pub;
                } else {
                    __builtin_amdgcn_s_sleep(1);
                    if ((++spins & 1023) == 0 && rt() - t0 > 2 * DEADLINE) break;
                }
            }
        }
        STAMP(9);
    } else {
        const int wm = wave >> 1, wn = wave & 1;
        if (shortrow && wm == 4) return;
        const float scale = P.sx[0] * P.sw[0];
        v16f acc[2][4];
#pragma unroll
        for (int i = 0; i < 2; ++i)
#pragma unroll
            for (int j = 0; j < 4; ++j)
#pragma unroll
                for (int r = 0; r < 16; ++r) acc[i][j][r] = 0.f;
        const LAS char* xa = L + wm * UNIT + lane * 16;
        const LAS char* wb = L + XB + wn * 8192 + lane * 16;
        volatile LAS unsigned* fullp = (volatile LAS unsigned*)(L + L_FULL);
        volatile LAS unsigned* freep = (volatile LAS unsigned*)(L + L_FREE + wave * 4);
        unsigned fw = 0;
        for (unsigned pos = 0; pos < 64; ++pos) {
            if (rfl(fw) <= pos) {
                unsigned spins = 0;
                while (rfl(*fullp) <= pos) {
                    __builtin_amdgcn_s_sleep(1);
                    if ((++spins & 1023) == 0 && rt() - t0 > 3 * DEADLINE) break;
                }
            }
            asm volatile("" ::: "memory");
            const unsigned so = (pos & (NSLOT - 1)) * SLOTB;
            const LAS char* xs = xa + so;
            const LAS char* wsl = wb + so;
            v8i x0 = ldfrag(xs, 0, 1024);
            v8i w0 = ldfrag(wsl, 0, 4096);
            v8i x1 = ldfrag(xs, 2048, 3072);
            v8i w1 = ldfrag(wsl, 1024, 5120);
            acc[0][0] = MFMA(x0, w0, acc[0][0]);
            acc[1][0] = MFMA(x1, w0, acc[1][0]);
            v8i w2 = ldfrag(wsl, 2048, 6144);
            acc[0][1] = MFMA(x0, w1, acc[0][1]);
            acc[1][1] = MFMA(x1, w1, acc[1][1]);
            v8i w3 = ldfrag(wsl, 3072, 7168);
            acc[0][2] = MFMA(x0, w2, acc[0][2]);
            acc[1][2] = MFMA(x1, w2, acc[1][2]);
            asm volatile("s_waitcnt lgkmcnt(0)" : "+v"(w3) :: "memory");
            *freep = pos + 1;
            fw = *fullp;
            asm volatile("" : "+v"(w3) :: "memory");
            acc[0][3] = MFMA(x0, w3, acc[0][3]);
            acc[1][3] = MFMA(x1, w3, acc[1][3]);
        }
        if (wave == 0) STAMP(16);
        const bool bad = rfl(*(volatile LAS unsigned*)(L + L_ABORT)) != 0;
        const float s = bad ? __builtin_nanf("") : scale;
        auto ro = mkrsrc((char*)P.out + ((size_t)(ti * 320 + wm * 64) * MN + tj * 256 + wn * 128) * 4, 64u * MN * 4u);
        const unsigned vo = (lane >> 5) * 4 * (MN * 4) + (lane & 31) * 16;
#pragma unroll
        for (int mb = 0; mb < 2; ++mb)
#pragma unroll
            for (int r = 0; r < 16; ++r) {
                v4f ov = {acc[mb][0][r] * s, acc[mb][1][r] * s, acc[mb][2][r] * s, acc[mb][3][r] * s};
                __builtin_amdgcn_raw_buffer_store_b128(__builtin_bit_cast(v4u, ov), ro,
                                                       vo + (mb * 32 + (r & 3) + 8 * (r >> 2)) * (MN * 4), 0, 16);
            }
    }
}

extern "C" void kernel_launch(int rank, int nranks, ncclComm_t comm,
                              hipStream_t stream,
                              void* const* d_in, const int* in_sizes, int n_in,
                              void* d_out, int out_size,
                              const int* in_dtypes, int out_dtype,
                              void* const* d_out_all,
                              void* const* d_ws_all, size_t ws_bytes) {
    if (nranks != NR || (unsigned)rank >= (unsigned)NR || n_in != 4 || ws_bytes < WS_NEED ||
        in_sizes[0] != MN * KS || in_sizes[1] != MN * KS || out_size != MN * MN) {
        fprintf(stderr, "kernel_launch: unexpected problem shape\n");
        return;
    }
    Params P;
    P.x = (const unsigned char*)d_in[0];
    P.w = (const unsigned char*)d_in[1];
    P.sx = (const float*)d_in[2];
    P.sw = (const float*)d_in[3];
    P.out = (float*)d_out;
    for (int p = 0; p < NR; ++p) P.ws[p] = (char*)d_ws_all[p];
    P.rank = rank;
    hipLaunchKernelGGL(gemm_ag_kernel, dim3(NWG), dim3(NTHREADS), 0, stream, P);
    HIP_CHECK(hipGetLastError());
}
```

```cpp
#include <hip/hip_runtime.h>
#include <rccl/rccl.h>
#include <cstdio>

#define HIP_CHECK(expr) do { hipError_t _e = (expr); if (_e != hipSuccess) { \
    fprintf(stderr, "HIP error %s at %s:%d\n", hipGetErrorString(_e), __FILE__, __LINE__); \
    return; } } while (0)

typedef int v8i __attribute__((ext_vector_type(8)));
typedef int v4i __attribute__((ext_vector_type(4)));
typedef unsigned v4u __attribute__((ext_vector_type(4)));
typedef float v16f __attribute__((ext_vector_type(16)));
typedef float v4f __attribute__((ext_vector_type(4)));
typedef unsigned long long u64;
#define LAS __attribute__((address_space(3)))
#define GAS __attribute__((address_space(1)))

namespace {
constexpr int NR = 8;
constexpr int MN = 4096;
constexpr int KS = 512;
constexpr int NTHREADS = 768;
constexpr int NWG = 256;
constexpr unsigned UNIT = 4096;
constexpr unsigned QB = 64 * UNIT;
constexpr unsigned OPB = 64 * QB;
constexpr unsigned GB = 2 * OPB;
constexpr size_t OFF_CTR = 0;
constexpr size_t OFF_TMO = 32768;
constexpr size_t OFF_TK = 36864;
constexpr size_t OFF_SF = 65536;
constexpr size_t OFF_RFX = 131072;
constexpr size_t OFF_RFW = 131072 + 32768;
constexpr size_t OFF_DBG = 262144;
constexpr size_t OFF_G = 1u << 20;
constexpr size_t WS_NEED = OFF_G + 2 * (size_t)GB;
constexpr unsigned L_WA = 10 * UNIT;
constexpr unsigned L_MIR = 38 * UNIT;
constexpr unsigned L_CTL = L_MIR + 3072;
constexpr unsigned L_FREE = L_CTL;
constexpr unsigned L_STG = L_CTL + 128;
constexpr unsigned L_ORDER = L_CTL + 256;
constexpr unsigned L_WQ = L_CTL + 640;
constexpr unsigned L_D = L_CTL + 1024;
constexpr unsigned L_CALL = L_CTL + 2040;
constexpr unsigned L_ABORT = L_CTL + 2044;
constexpr unsigned LDS_BYTES = L_CTL + 2048;
constexpr unsigned XB = 5;
constexpr unsigned DEADLINE = 50000000u;
#define CL 2
#define CX 0
#define PCLK 800
#define HPRIO 3
#define EPS 2
#define NP 12
#define PE NP
#define PD 0
#define SD 8
#define SU (NP + NWL)
#define NB 2
#define PR 0
#define XR 4
#define FPRIO 2
#define EPA 0
#define SPM 3
#define BPTH 2
#define FSL 1
#define BPRIO 1
#define ANS 1
#define EPM 1
#define EFM 2
#define EPIE "nt"
#define EPIF "sc1"
#define NWL 20
#define REV 1
#define PTH 3
#define LG4 1
#define NCC 48
#define MAP 1
#define RS 4

struct Params {
    const unsigned char* x;
    const unsigned char* w;
    const float* sx;
    const float* sw;
    float* out;
    int rank;
    char* ws[NR];
};

__device__ __forceinline__ __amdgpu_buffer_rsrc_t mkrsrc(const void* p, unsigned bytes) {
    return __builtin_amdgcn_make_buffer_rsrc((void*)p, 0, bytes, 0x00020000);
}
__device__ __forceinline__ u64 rt() { return __builtin_amdgcn_s_memrealtime(); }
__device__ __forceinline__ unsigned rfl(unsigned v) { return __builtin_amdgcn_readfirstlane(v); }
__device__ __forceinline__ unsigned ld_agent(const char* p) {
    return __hip_atomic_load((const GAS unsigned*)p, __ATOMIC_RELAXED, __HIP_MEMORY_SCOPE_AGENT);
}
__device__ __forceinline__ unsigned ld_sys(const char* p) {
    return __hip_atomic_load((const GAS unsigned*)p, __ATOMIC_RELAXED, __HIP_MEMORY_SCOPE_SYSTEM);
}
__device__ __forceinline__ void st_agent(char* p, unsigned v) {
    __hip_atomic_store((GAS unsigned*)p, v, __ATOMIC_RELAXED, __HIP_MEMORY_SCOPE_AGENT);
}
__device__ __forceinline__ void st_sys(char* p, unsigned v) {
    __hip_atomic_store((GAS unsigned*)p, v, __ATOMIC_RELAXED, __HIP_MEMORY_SCOPE_SYSTEM);
}
#define WAITV(n) asm volatile("s_waitcnt vmcnt(" #n ")" ::: "memory")
#define WAITL0() asm volatile("s_waitcnt lgkmcnt(0)" ::: "memory")

#define STAMP(idx) do { } while (0)

#define DMAF "sc1"
#define GAPLD(r) "buffer_load_dword v167, off, " r ", 0\n\t"
#define LGAP(r) ""
template <bool G4 = false>
__device__ __forceinline__ void dma4k(__amdgpu_buffer_rsrc_t r, __amdgpu_buffer_rsrc_t rq, unsigned voff, unsigned soff, unsigned lds) {
    asm volatile(
        "s_mov_b32 m0, %0\n\t"
        "s_nop 3\n\t"
        "buffer_load_dwordx4 %1, %2, %3 offen " DMAF " lds\n\t"
        "buffer_load_dwordx4 %1, %2, %3 offen offset:1024 " DMAF " lds\n\t"
        ".if %5 == 0\n\t" GAPLD("%4") ".endif\n\t"
        "buffer_load_dwordx4 %1, %2, %3 offen offset:2048 " DMAF " lds\n\t"
        "buffer_load_dwordx4 %1, %2, %3 offen offset:3072 " DMAF " lds\n\t"
        GAPLD("%4")
        :: "s"(lds), "v"(voff), "s"(r), "s"(soff), "s"(rq), "n"(G4) : "memory", "v167");
}

__device__ __forceinline__ u64 ticket(GAS u64* p, int lane) {
    u64 r = 0, one = 1;
    if (lane == 0) asm volatile("global_atomic_add_x2 %0, %1, %2, off sc0" : "=&v"(r) : "v"(p), "v"(one) : "memory");
    return r;
}

#define MF(c, a, b) "v_mfma_f32_32x32x64_f8f6f4 " c ", " a ", " b ", " c "\n\t"

#define DM(so, off) "buffer_load_dwordx4 %[voff], %[rg], " so " offen " off " " DMAF " lds\n\t"

__device__ __forceinline__ unsigned unit_of(unsigned n, unsigned link) {
    const unsigned c = CL + (link < CX), na = 64 + (7 - c) * 128, nb = na + (c - 1) * 102, nc = nb + 102 - 13 * NB, nd = nc + 64;
    unsigned j, op, u;
    if (n < 64) {
        j = 0; op = 0; u = n;
    } else if (n < na) {
        unsigned m = n - 64;
        op = ((m >> 2) & 1) ^ 1;
        j = (m >> 7) + 1 - op;
        u = ((m >> 3) & 15) * 4 + (m & 3);
    } else if (n < nb) {
        unsigned m = n - na, jj = m / 102, i = m - jj * 102;
        unsigned g = i >> 3, r = (i & 7) + (i >= 98);
        op = r >= 3;
        j = 8 - c + jj - op;
        u = 5 * g + (op ? r - 3 : r + 2);
    } else if (n < nc) {
        unsigned m = n - nb, g = m / (8 - NB), r = m - g * (8 - NB);
        r += g == 12 && r >= 2 - NB;
        op = r >= 3 - NB;
        j = 7 - op;
        u = 5 * g + (op ? r - (3 - NB) : r + 2 + NB);
    } else if (n < nd) {
        j = 7; op = 1; u = n - nc;
    } else {
        unsigned m = n - nd;
        if (NB == 1) m += m < 13 ? m : 13;
        unsigned jj = m / 26, i = m - jj * 26 + 2 * ((link * 13) / 7);
        if (i >= 26) i -= 26;
        j = jj ? 8 - jj : 7;
        op = 0;
        u = 5 * (i >> 1) + (i & 1) + (jj ? 0 : 2);
    }
    if (op) u = (u + 4 * ((link * 16) / 7)) & 63;
    return (op << 9) | (((j + link) & 7) << 6) | u;
}
struct SeqT { unsigned v[64]; constexpr SeqT() : v() { for (unsigned i = 0; i < 64; ++i) v[i] = i * UNIT; } };
__device__ const SeqT SEQ[1] = {SeqT()};
__device__ __forceinline__ unsigned rf_off(unsigned op, unsigned q, unsigned u) {
    unsigned ti = (u * 205u) >> 10, r = u - ti * 5, e = r >> 2;
    return op ? (unsigned)OFF_RFW + (((u >> 2) * 64 + q) * 4 + (u & 3)) * 4
              : (unsigned)OFF_RFX + (((ti * 2 + e) * 64 + q) * 4 + (r & 3)) * 4;
}
__device__ __forceinline__ void dma1k(__amdgpu_buffer_rsrc_t r, unsigned voff, unsigned soff, unsigned lds) {
    asm volatile(
        "s_mov_b32 m0, %0\n\t"
        "s_nop 3\n\t"
        "buffer_load_dwordx4 %1, %2, %3 offen sc0 sc1 lds"
        :: "s"(lds), "v"(voff), "s"(r), "s"(soff) : "memory");
}
__device__ __forceinline__ void dma4(__amdgpu_buffer_rsrc_t r, unsigned soff, unsigned lds) {
    asm volatile(
        "s_mov_b32 m0, %0\n\t"
        "s_mov_b64 exec, 1\n\t"
        "s_nop 3\n\t"
        "buffer_load_dword off, %1, %2 lds\n\t"
        "s_mov_b64 exec, -1"
        :: "s"(lds), "s"(r), "s"(soff) : "memory");
}
}

__global__ void __launch_bounds__(NTHREADS, 3) gemm_ag_kernel(Params P) {
    __shared__ __attribute__((aligned(16))) char smem[LDS_BYTES];
    LAS char* L = (LAS char*)smem;
    const int tid = threadIdx.x;
    const int wave = rfl(tid >> 6);
    const int lane = tid & 63;
    const int b = blockIdx.x;
    const int rank = P.rank;
    char* myws = P.ws[0];

    {
        const int e = wave - 8;
        const unsigned sop = b >> 7, st = (b >> 4) & 7, sblk = b & 15;
        v4u o[4];
        char* pl = nullptr;
        if (e >= 0) {
            STAMP(e);
            pl = P.ws[lane & 7];
            if (SD && (sop || st == 7 || sblk * 4 >= SU)) __builtin_amdgcn_s_sleep(SD);
            asm volatile("" ::: "memory");
            if (sop == 0) {
                auto rx = mkrsrc(P.x, MN * KS);
                unsigned base = (sblk * 256 + e * 64 + (lane & 31)) * KS + st * 64 + (lane >> 5) * 16;
                o[0] = __builtin_amdgcn_raw_buffer_load_b128(rx, base, 0, 0);
                o[1] = __builtin_amdgcn_raw_buffer_load_b128(rx, base + 32, 0, 0);
                o[2] = __builtin_amdgcn_raw_buffer_load_b128(rx, base + 32 * KS, 0, 0);
                o[3] = __builtin_amdgcn_raw_buffer_load_b128(rx, base + 32 * KS + 32, 0, 0);
            } else {
                auto rw = mkrsrc(P.w, MN * KS);
                unsigned base = (st * 64 + (e & 1) * 32 + (lane >> 5) * 16) * MN + sblk * 256 + (e >> 1) * 128 + (lane & 31) * 4;
                unsigned d[16];
#pragma unroll
                for (int i = 0; i < 16; ++i) d[i] = __builtin_amdgcn_raw_buffer_load_b32(rw, base, i * MN, 0);
#pragma unroll
                for (int g = 0; g < 4; ++g) {
                    unsigned t0 = __builtin_amdgcn_perm(d[4 * g + 1], d[4 * g], 0x05010400u);
                    unsigned t1 = __builtin_amdgcn_perm(d[4 * g + 1], d[4 * g], 0x07030602u);
                    unsigned t2 = __builtin_amdgcn_perm(d[4 * g + 3], d[4 * g + 2], 0x05010400u);
                    unsigned t3 = __builtin_amdgcn_perm(d[4 * g + 3], d[4 * g + 2], 0x07030602u);
                    o[0][g] = __builtin_amdgcn_perm(t2, t0, 0x05040100u);
                    o[1][g] = __builtin_amdgcn_perm(t2, t0, 0x07060302u);
                    o[2][g] = __builtin_amdgcn_perm(t3, t1, 0x05040100u);
                    o[3][g] = __builtin_amdgcn_perm(t3, t1, 0x07060302u);
                }
            }
        } else if (wave == 0) {
            if (lane == 0) {
                unsigned c = ld_agent(myws + OFF_CTR + b * 128) + 1;
                st_agent(myws + OFF_CTR + b * 128, c);
                *(volatile LAS unsigned*)(L + L_CALL) = c;
                *(volatile LAS unsigned*)(L + L_ABORT) = 0;
            }
        } else if (wave <= 5) {
            if (tid < 64 + 319) *(volatile LAS v4u*)(L + L_MIR + (tid - 64) * 16) = v4u{0, 0, 0, 0};
        }
        WAITL0();
        __builtin_amdgcn_s_barrier();
        if (e >= 0) {
            if (e == 0) STAMP(29);
            const unsigned call = rfl(*(volatile LAS unsigned*)(L + L_CALL));
            auto rg = mkrsrc(myws + OFF_G + (size_t)(call & 1) * GB, GB);
            const unsigned q = rank * 8 + st, u = sblk * 4 + e;
            unsigned off = sop * OPB + q * QB + u * UNIT + lane * 16;
#pragma unroll
            for (int i = 0; i < 4; ++i) __builtin_amdgcn_raw_buffer_store_b128(o[i], rg, off + i * 1024, 0, 17);
            static_assert(NP + NWL <= 64 && PE <= NP, "");
            const bool psh = sop == 0 && st < 7 && u < NP;
            char* pp = P.ws[(7 - st) & 7];
            auto rp = mkrsrc(pp + OFF_G + (size_t)(call & 1) * GB, GB);
            if (psh && u < PE) {
#pragma unroll
                for (int i = 0; i < 4; ++i) __builtin_amdgcn_raw_buffer_store_b128(o[i], rp, off + i * 1024, 0, 17);
                WAITV(4);
            } else {
                WAITV(0);
            }
            if (e == 0) STAMP(31);
            if (sop == 0 && u - NP < NWL && lane + st == 7 && lane != 0) st_sys(pl + OFF_SF + 32768 + (rank * 64 + u - NP) * 64, call);
            unsigned old = 0;
            if (lane == 0) {
                old = __hip_atomic_fetch_add((LAS unsigned*)(L + L_STG), 1u, __ATOMIC_RELAXED, __HIP_MEMORY_SCOPE_WORKGROUP);
                st_agent(myws + rf_off(sop, q, u), call);
            }
            if (rfl(old) == 3 && lane < NR && lane != 0) st_sys(pl + OFF_SF + (rank * 1024 + b * 4) * 4, call);
            if (psh) {
                if (u >= PE) {
                    if (PD) __builtin_amdgcn_s_sleep(PD);
#pragma unroll
                    for (int i = 0; i < 4; ++i) __builtin_amdgcn_raw_buffer_store_b128(o[i], rp, off + i * 1024, 0, 17);
                }
                WAITV(0);
                if (lane == 0) st_sys(pp + rf_off(0, q, u), call);
            }
            STAMP(4 + e);
        }
    }
    const unsigned call = rfl(*(volatile LAS unsigned*)(L + L_CALL));
    const unsigned par = call & 1;
    const unsigned t0 = (unsigned)rt();
    const int xcd = b & 7, idx = b >> 3;

    if (idx < 6) {
        const unsigned wi = wave * NCC + b;
        if (b >= NCC || wi >= 7 * NWL) return;
        const unsigned link = wi % 7, widx = wi / 7;
        auto rg = mkrsrc(myws + OFF_G + (size_t)par * GB, GB);
        const unsigned src = (rank + 1 + link) & 7;
        auto rs = mkrsrc(P.ws[1 + link] + OFF_G + (size_t)par * GB, GB);
        auto rsf = mkrsrc(myws + OFF_SF + src * 4096, 4096);
        GAS u64* tkp = (GAS u64*)(myws + OFF_TK + link * 128);
        const u64 tkbase = (u64)call << 12;
        const unsigned voff = lane * 16;
        auto unit_off = [&](unsigned id) { return (id >> 9) * OPB + (src * 8 + ((id >> 6) & 7)) * QB + (id & 63) * UNIT; };
        auto rf_addr = [&](unsigned id) { return myws + rf_off(id >> 9, src * 8 + ((id >> 6) & 7), id & 63); };
        auto ld_sf = [&](unsigned id, bool all) -> unsigned {
            if (!all) return __builtin_amdgcn_raw_buffer_load_b32(rsf, (id & ~3u) * 4, 0, 17);
            unsigned m = ~0u;
#pragma unroll
            for (int i = 0; i < 4; ++i) m = min(m, __builtin_amdgcn_raw_buffer_load_b32(rsf, voff + i * 1024, 0, 17));
            return m;
        };
        auto wait_sf = [&](unsigned id, bool all) -> bool {
            unsigned sp_ = 0;
            while (__ballot(ld_sf(id, all) < call) != 0) {
                if ((++sp_ & 15) == 0 && (rfl(ld_agent(myws + OFF_TMO)) == call || (unsigned)rt() - t0 > DEADLINE)) {
                    if (lane == 0) st_agent(myws + OFF_TMO, call);
                    return false;
                }
                __builtin_amdgcn_s_sleep(1);
            }
            return true;
        };
        if (lane == 0) __hip_atomic_fetch_max(tkp, tkbase, __ATOMIC_RELAXED, __HIP_MEMORY_SCOPE_AGENT);
        unsigned id = unit_of(NP + widx, link);
        bool alive = true;
        for (unsigned sp_ = 1; rfl(ld_sys(myws + OFF_SF + 32768 + (src * 64 + widx) * 64)) < call; ++sp_) {
            if ((sp_ & 15) == 0 && (rfl(ld_agent(myws + OFF_TMO)) == call || (unsigned)rt() - t0 > DEADLINE)) {
                if (lane == 0) st_agent(myws + OFF_TMO, call);
                alive = false;
                break;
            }
            __builtin_amdgcn_s_sleep(1);
        }
        STAMP(24 + wave);
        v4u v[4];
        unsigned so = unit_off(id);
        if (alive) {
#pragma unroll
            for (int i = 0; i < 4; ++i) v[i] = __builtin_amdgcn_raw_buffer_load_b128(rs, so + voff + i * 1024, 0, 17);
#pragma unroll
            for (int r = 1; r < RS; ++r) {
                const bool all = r == RS - 1;
                const unsigned id2 = unit_of(NP + widx + r * NWL, link);
                unsigned f = ld_sf(id2, all);
#pragma unroll
                for (int i = 0; i < 4; ++i) __builtin_amdgcn_raw_buffer_store_b128(v[i], rg, so + voff + i * 1024, 0, 16);
                bool flagged = false;
                if (__ballot(f < call) != 0) {
                    WAITV(0);
                    if (lane == 0) st_agent(rf_addr(id), call);
                    flagged = true;
                    if (!wait_sf(id2, all)) { alive = false; break; }
                }
                so = unit_off(id2);
#pragma unroll
                for (int i = 0; i < 4; ++i) v[i] = __builtin_amdgcn_raw_buffer_load_b128(rs, so + voff + i * 1024, 0, 17);
                if (!flagged) {
                    WAITV(4);
                    if (lane == 0) st_agent(rf_addr(id), call);
                }
                id = id2;
            }
        }
        if (alive) {
            u64 tk = ticket(tkp, lane);
            while (true) {
                id = rfl(id);
                so = rfl(so);
#pragma unroll
                for (int i = 0; i < 4; ++i) __builtin_amdgcn_raw_buffer_store_b128(v[i], rg, so + voff + i * 1024, 0, 16);
                asm volatile("s_waitcnt vmcnt(4)" : "+v"(tk) :: "memory");
                const unsigned n2 = rfl((unsigned)(tk - tkbase)) + RS * NWL + NP;
                if (n2 >= 1024) {
                    WAITV(0);
                    if (lane == 0) st_agent(rf_addr(id), call);
                    break;
                }
                const unsigned id2 = unit_of(n2, link);
                so = unit_off(id2);
#pragma unroll
                for (int i = 0; i < 4; ++i) v[i] = __builtin_amdgcn_raw_buffer_load_b128(rs, so + voff + i * 1024, 0, 17);
                asm volatile("" ::: "memory");
                tk = ticket(tkp, lane);
                WAITV(5);
                if (lane == 0) st_agent(rf_addr(id), call);
                id = id2;
            }
        }
        STAMP(40 + wave);
        return;
    }

    const int ti = MAP == 2 ? (idx < 30 ? (xcd >> 2) * 6 + ((idx - 6) >> 2) : 12) : idx < 30 ? (xcd >> 2) * 7 + ((idx - 6) >> 2) : 6;
    const int tj = (xcd & 3) * 4 + (idx < 30 ? (idx - 6) & 3 : (xcd >> 2) * 2 + (idx & 1));
    const bool shortrow = ti == 12;
    constexpr unsigned NL = 7 * CL + CX, NC = 64 - NL, EA = 57, EC = 71, NE = EC + NL;
    static_assert(NL >= 8 && NE <= 120 && CX == 0, "");
    const unsigned fmask = shortrow ? 0xffu : 0x3ffu;

    if (wave >= 10) {
        __builtin_amdgcn_s_setprio(HPRIO);
        auto rg = mkrsrc(myws + OFF_G + (size_t)par * GB, GB);
        auto rq = mkrsrc(SEQ, sizeof(SEQ));
        const unsigned voff = lane * 16;
        unsigned e = 0, spins = 0;
        if (wave == 10) {
            auto rf = mkrsrc(myws + OFF_RFX, 65536);
            const unsigned lk = ((lane >> 3) - rank - 1) & 7, lj = ((lane & 7) - lk) & 7;
            const u64 split = __ballot(lk != 7 && lj >= 8 - CL - (lk < CX));
            const u64 lastk = __ballot(lk != 7 && lj == 7);
            u64 todo = ~split, pend = 0;
            unsigned tpoll = (unsigned)__builtin_amdgcn_s_memtime() - PCLK;
            unsigned lim = call;
            auto timeout = [&]() {
                if (rfl(ld_agent(myws + OFF_TMO)) == call || (unsigned)rt() - t0 > DEADLINE) {
                    st_agent(myws + OFF_TMO, call);
                    *(volatile LAS unsigned*)(L + L_ABORT) = 1;
                    lim = 0;
                }
                return (unsigned)rt() - t0 > 2 * DEADLINE;
            };
            while (e < 64) {
                const unsigned fv = max(*(volatile LAS unsigned*)(L + L_FREE + lane * 4), L_D);
                if ((pend & (pend - 1)) == 0) {
                    const unsigned now = (unsigned)__builtin_amdgcn_s_memtime();
                    v4u a = *(volatile LAS v4u*)(L + L_MIR + voff);
                    unsigned d = *(volatile LAS unsigned*)(L + L_MIR + 1024 + voff);
                    v4u c = *(volatile LAS v4u*)(L + L_MIR + 2048 + voff);
                    unsigned m = min(min(c.x, c.y), min(c.z, c.w));
                    if (!shortrow) m = min(m, d);
                    if (e < EA || NB == 1) m = min(m, a.w);
                    if (e < EA) m = min(m, a.z);
                    if (e < NC) m = min(m, min(a.x, a.y));
                    pend = __ballot(m >= lim) & todo;
                    if (now - tpoll >= PCLK) {
                        tpoll = now;
                        WAITV(60);
                        dma1k(rf, voff, ti * 2048 + 1024, L_MIR + 1024);
                        dma1k(rf, voff, 32768 + tj * 1024, L_MIR + 2048);
                        dma1k(rf, voff, ti * 2048, L_MIR);
                    }
                }
                const unsigned need = L_D + 8 * (e <= EA ? e - 3 : max(EA, e - 4));
                const bool fr = ((unsigned)__ballot(fv >= need) & fmask) == fmask;
                if (pend == 0 || !fr) {
                    if ((++spins & 255) == 0 && timeout()) break;
                    __builtin_amdgcn_s_sleep(1);
                    continue;
                }
                const unsigned q = __builtin_ctzll(pend);
                pend &= pend - 1;
                todo &= ~(1ull << q);
                *(volatile LAS unsigned*)(L + L_ORDER + e * 4) = q + 1;
                const unsigned sx = q * QB + ti * 5 * UNIT;
                const unsigned lds = e < EA ? (e & 3) * (9 * UNIT) : (e < EA + 5 ? e - EA : e - EA - 5) * (2 * UNIT) - (2 + NB) * UNIT;
                WAITV(32);
                if (e < EA) dma4k(rg, rq, voff, sx + 2 * UNIT, lds + 2 * UNIT);
                if (e < EA || NB == 1) dma4k(rg, rq, voff, sx + 3 * UNIT, lds + 3 * UNIT);
                if (!shortrow) dma4k(rg, rq, voff, sx + 4 * UNIT, lds + 4 * UNIT);
                if (e < NC) {
                    dma4k(rg, rq, voff, sx, lds);
                    dma4k(rg, rq, voff, sx + UNIT, lds + UNIT);
                }
                dma4(rq, (lds + XB * UNIT) >> 10, L_D + e * 8);
                ++e;
                if (e == NC) { todo = split & ~lastk; pend = 0; }
                if (e == EA) { todo = lastk; pend = 0; }
            }
            const unsigned qx = (REV && (tj & 1)) ? 63 : 0;
            const u64 lk7 = qx ? __builtin_bitreverse64(lastk) : lastk;
            u64 hist = 0;
            unsigned need = L_D + 8 * (e - 4), slot = 4 * UNIT, cnt;
#pragma nounroll
            for (unsigned ph = e == 64 ? 0 : 2; ph < 2; ++ph) {
                todo = lk7;
                u64 rem = ph == 0 ? lk7 : qx ? __builtin_bitreverse64(split) : split;
                const unsigned nend = L_D + 8 * ((ph ? NE : EC) - 4);
                while (need < nend) {
                    unsigned fv, m0v, m1v, wq, sq, sa;
                    u64 pk;
                    spins = 0;
                    asm volatile(
                        "s_branch 3f\n"
                        "1:\n\t"
                        "s_waitcnt lgkmcnt(0)\n\t"
                        "v_min_u32 %[m0v], %[m0v], %[m1v]\n\t"
                        "v_add_u32 %[fv], %[xr], %[fv]\n\t"
                        "v_cmp_le_u32 vcc, %[need], %[fv]\n\t"
                        "s_or_b32 %[sa], vcc_lo, %[nfm]\n\t"
                        "v_cmp_ne_u32 %[pk], 0, %[wq]\n\t"
                        "v_cmp_le_u32 vcc, %[lim], %[m0v]\n\t"
                        "s_orn2_b64 %[pk], %[pk], %[lk7]\n\t"
                        "s_and_b64 vcc, vcc, %[rem]\n\t"
                        "s_and_b64 %[pk], %[pk], %[todo]\n\t"
                        "s_and_b64 %[pk], %[pk], vcc\n\t"
                        "s_cbranch_scc0 2f\n\t"
                        "s_cmp_eq_u32 %[sa], -1\n\t"
                        "s_cbranch_scc0 6f\n\t"
                        "s_ff1_i32_b64 %[sq], %[pk]\n\t"
                        "s_bcnt1_i32_b64 %[cnt], vcc\n\t"
                        "s_cmp_eq_u64 vcc, %[rem]\n\t"
                        "s_cselect_b32 %[cnt], 64, %[cnt]\n\t"
                        "s_mov_b32 m0, %[slot]\n\t"
                        "v_mov_b32 %[fv], %[need]\n\t"
                        "s_lshl_b64 %[pk], 1, %[sq]\n\t"
                        "s_and_b64 exec, %[pk], %[lk7]\n\t"
                        "s_xor_b32 %[sa], %[sq], %[qx]\n\t"
                        "ds_write_b32 %[fv], %[wq] offset:36\n\t"
                        "s_mov_b32 exec_lo, %[x1]\n\t"
                        "s_mov_b32 exec_hi, %[x1]\n\t"
                        "s_lshl_b32 %[sa], %[sa], 18\n\t"
                        "s_add_u32 %[sa], %[sa], %[xo]\n\t"
                        "s_waitcnt vmcnt(50)\n\t"
                        DM("%[sa]", "")
                        "s_bitset0_b64 %[todo], %[sq]\n\t"
                        "s_bitset0_b64 %[rem], %[sq]\n\t"
                        "s_add_u32 vcc_lo, %[sq], %[dir]\n\t"
                        "s_and_b32 vcc_lo, vcc_lo, 7\n\t"
                        DM("%[sa]", "offset:1024") LGAP("%[rq]")
                        "s_and_b32 %[sq], %[sq], 56\n\t"
                        "s_or_b32 %[sq], %[sq], vcc_lo\n\t"
                        "s_lshl_b32 vcc_hi, %[sq], 16\n\t"
                        "s_mov_b32 vcc_lo, 0\n\t"
                        DM("%[sa]", "offset:2048")
                        "s_lshr_b64 %[hist], %[hist], 8\n\t"
                        "s_or_b64 vcc, vcc, %[hist]\n\t"
                        "s_mov_b64 %[hist], vcc\n\t"
                        "s_lshl_b64 %[pk], 1, vcc_lo\n\t"
                        "s_cmp_eq_u64 %[todo], 0\n\t"
                        "s_cselect_b64 %[todo], %[pk], %[todo]\n\t"
                        DM("%[sa]", "offset:3072") GAPLD("%[rq]")
                        "s_add_u32 m0, m0, 4096\n\t"
                        "s_mov_b64 exec, -1\n\t"
                        "s_add_u32 %[sa], %[sa], 4096\n\t"
                        "s_lshr_b32 %[sq], %[slot], 10\n\t"
                        "s_add_u32 %[sq], %[sq], %[xi]\n\t"
                        DM("%[sa]", "")
                        DM("%[sa]", "offset:1024") LGAP("%[rq]")
                        "ds_read_b32 %[fv], %[vctl]\n\t"
                        "ds_read_b32 %[m0v], %[vmir]\n\t"
                        "ds_read_b32 %[m1v], %[vmir] offset:4\n\t"
                        "ds_read_b32 %[wq], %[vwq]\n\t"
                        DM("%[sa]", "offset:2048")
                        DM("%[sa]", "offset:3072") GAPLD("%[rq]")
                        "s_add_u32 m0, %[need], 32\n\t"
                        "s_mov_b64 exec, 1\n\t"
                        "s_add_u32 %[slot], %[slot], 8192\n\t"
                        "s_cmp_eq_u32 %[slot], %[ring]\n\t"
                        "s_cselect_b32 %[slot], 0, %[slot]\n\t"
                        "buffer_load_dword off, %[rq], %[sq] lds\n\t"
                        "s_mov_b32 m0, %[lmir]\n\t"
                        "s_mov_b64 exec, -1\n\t"
                        "s_add_u32 %[need], %[need], 8\n\t"
                        "s_cmp_lt_u32 %[need], %[nend]\n\t"
                        "s_cbranch_scc0 4f\n\t"
                        "s_cmp_gt_u32 %[cnt], %[pth]\n\t"
                        "s_cbranch_scc1 1b\n\t"
                        "buffer_load_dwordx4 %[voff], %[rf], %[fo] offen sc0 sc1 lds\n\t"
                        "s_branch 1b\n"
                        "6:\n\t"
                        "s_add_u32 %[spins], %[spins], 1\n\t"
                        "s_cmp_eq_u32 %[spins], 256\n\t"
                        "s_cbranch_scc0 5f\n\t"
                        "s_branch 4f\n"
                        "2:\n\t"
                        "s_sleep 1\n\t"
                        "s_add_u32 %[spins], %[spins], 1\n\t"
                        "s_cmp_eq_u32 %[spins], 256\n\t"
                        "s_cbranch_scc1 4f\n\t"
                        "s_and_b32 %[sa], %[spins], %[spm]\n\t"
                        "s_cbranch_scc1 5f\n"
                        "3:\n\t"
                        "s_mov_b32 m0, %[lmir]\n\t"
                        "s_waitcnt vmcnt(62)\n\t"
                        "s_nop 1\n\t"
                        "buffer_load_dwordx4 %[voff], %[rf], %[fo] offen sc0 sc1 lds\n"
                        "5:\n\t"
                        "ds_read_b32 %[fv], %[vctl]\n\t"
                        "ds_read_b32 %[m0v], %[vmir]\n\t"
                        "ds_read_b32 %[m1v], %[vmir] offset:4\n\t"
                        "ds_read_b32 %[wq], %[vwq]\n\t"
                        "s_branch 1b\n"
                        "4:\n\t"
                        "s_waitcnt lgkmcnt(0)"
                        : [slot] "+&s"(slot), [todo] "+&s"(todo), [need] "+&s"(need), [spins] "+&s"(spins), [rem] "+&s"(rem), [hist] "+&s"(hist),
                          [fv] "=&v"(fv), [m0v] "=&v"(m0v), [m1v] "=&v"(m1v), [wq] "=&v"(wq),
                          [sq] "=&s"(sq), [sa] "=&s"(sa), [cnt] "=&s"(cnt), [pk] "=&s"(pk)
                        : [voff] "v"(voff), [vctl] "v"(L_FREE + lane * 4), [vmir] "v"(L_MIR + (lane ^ qx) * 16 + (ph ? 0 : 8)),
                          [vwq] "v"(L_WQ + (lane ^ qx) * 4), [qx] "s"(qx), [lk7] "s"(lk7), [dir] "s"(qx ? 1 : 7), [pth] "s"(ph ? PTH : BPTH), [spm] "n"(SPM), [xr] "n"(XR), [rg] "s"(rg), [rq] "s"(rq),
                          [rf] "s"(rf), [lim] "s"(lim), [nfm] "s"(~fmask), [xo] "s"((ti * 5 + (ph ? 0 : NB)) * UNIT), [x1] "s"(rfl(NB == 2 ? ~0u : 0u - ph)),
                          [xi] "s"((ph ? XB : XB - NB) * 4), [fo] "s"(ti * 2048), [ring] "n"(L_WA), [lmir] "n"(L_MIR), [nend] "s"(nend)
                        : "memory", "scc", "vcc", "v167");
                    if (need < nend && timeout()) { ph = 2; break; }
                }
            }
        } else {
            unsigned v = *(volatile LAS unsigned*)(L + L_ORDER);
            u64 qa = 0;
            auto run = [&](unsigned upto) __attribute__((always_inline)) {
            while (e < upto) {
                const unsigned q1 = rfl(v);
                if (q1 == 0) {
                    if ((++spins & 1023) == 0 && (unsigned)rt() - t0 > 2 * DEADLINE) break;
                    __builtin_amdgcn_s_sleep(1);
                    v = *(volatile LAS unsigned*)(L + L_ORDER + e * 4);
                    continue;
                }
                const unsigned sw = OPB + (q1 - 1) * QB + tj * 4 * UNIT;
                const unsigned lds = e < EA ? (e & 3) * (9 * UNIT) + 5 * UNIT : L_WA + (e - EA) * (4 * UNIT);
                WAITV(38);
                dma4k<LG4>(rg, rq, voff, sw, lds);
                dma4k<LG4>(rg, rq, voff, sw + UNIT, lds + UNIT);
                dma4k<LG4>(rg, rq, voff, sw + 3 * UNIT, lds + 3 * UNIT);
                v = *(volatile LAS unsigned*)(L + L_ORDER + e * 4 + 4);
                dma4k<LG4>(rg, rq, voff, sw + 2 * UNIT, lds + 2 * UNIT);
                dma4(rq, lds >> 10, L_D + e * 8 + 4);
                if (e >= EA) {
                    dma4(rq, lds >> 10, L_WQ + (q1 - 1) * 4);
                    qa |= (u64)(q1 - 1) << (8 * (e - EA));
                }
                ++e;
            }
            };
            auto prev_w = [&](unsigned a) {
                const unsigned q = (unsigned)(qa >> (8 * a)) & 63;
                return OPB + ((q & 56) | ((q - 1) & 7)) * QB + tj * 4 * UNIT;
            };
            run(64);
            e = e == 64 ? EC : NE;
            __builtin_amdgcn_s_setprio(FPRIO);
            while (e < NE - 7) {
                const unsigned fv = *(volatile LAS unsigned*)(L + L_FREE + lane * 4);
                const unsigned wv = *(volatile LAS unsigned*)(L + L_D + e * 8 + 4);
                if (((unsigned)__ballot(fv >= L_D + e * 8 + 8) & fmask) != fmask) {
                    if ((++spins & 1023) == 0 && (unsigned)rt() - t0 > 2 * DEADLINE) break;
                    if (FSL) __builtin_amdgcn_s_sleep(FSL);
                    continue;
                }
                const unsigned a = min((rfl(wv) - L_WA) >> 14, 6u), lds = L_WA + a * (4 * UNIT);
                const unsigned sw = prev_w(a);
                qa ^= (u64)(((qa >> (8 * a)) ^ ((sw - OPB) / QB)) & 63) << (8 * a);
#pragma unroll
                for (int i = 0; i < 4; ++i) {
                    WAITV(57);
                    dma4k<LG4>(rg, rq, voff, sw + i * UNIT, lds + i * UNIT);
                }
                dma4(rq, lds >> 10, L_D + e * 8 + 60);
                ++e;
            }
        }
        WAITV(0);
        STAMP(wave - 1);
    } else {
        const int wm = wave >> 1, wn = wave & 1;
        if (shortrow && wm == 4) return;
        if (BPRIO && wave >= 4 + 2 * NB) __builtin_amdgcn_s_setprio(2);
        else if (BPRIO && wave >= 4) __builtin_amdgcn_s_setprio(1);
        const float scale = __builtin_bit_cast(float, rfl(__builtin_bit_cast(unsigned, P.sx[0] * P.sw[0])));
        unsigned xs, wsl, vl = lane * 16;
        const unsigned freep = L_FREE + wave * 4;
        auto ro = mkrsrc((char*)P.out + ((size_t)(ti * 320 + wm * 64) * MN + tj * 256 + wn * 128) * 4, 64u * MN * 4u);
        unsigned pos = L_D, end = L_D + 8 * (wave < 4 ? NC : wave < 4 + 2 * NB ? EA : 64);
        const unsigned last = L_D + 8 * (wave < 4 ? NE : wave < 4 + 2 * NB ? EC : 64), jump = L_D + 8 * (wave < 4 ? EC : 64);
        {
            unsigned lim, lim2, spins = 0;
            asm volatile(
                ".set gk_i, 0\n\t"
                ".rept 128\n\t"
                "v_mov_b32 v[gk_i], 0\n\t"
                ".set gk_i, gk_i + 1\n\t"
                ".endr\n"
                "5:\n\t"
                "v_mov_b32 v160, %[pos]\n\t"
                "ds_read_b64 v[160:161], v160\n\t"
                "s_waitcnt lgkmcnt(0)\n\t"
                "v_readfirstlane_b32 %[lim], v160\n\t"
                "v_readfirstlane_b32 %[lim2], v161\n\t"
                "v_add3_u32 %[xs], v160, %[vl], %[xo]\n\t"
                "v_add3_u32 %[ws], v161, %[vl], %[wo]\n\t"
                "s_min_u32 %[lim], %[lim], %[lim2]\n\t"
                "s_cmp_lg_u32 %[lim], 0\n\t"
                "s_cbranch_scc1 6f\n\t"
                "s_sleep 1\n\t"
                "s_add_u32 %[spins], %[spins], 1\n\t"
                "s_and_b32 %[lim], %[spins], 1023\n\t"
                "s_cbranch_scc1 5b\n\t"
                "s_memrealtime vcc\n\t"
                "s_waitcnt lgkmcnt(0)\n\t"
                "s_sub_u32 %[lim], vcc_lo, %[t0]\n\t"
                "s_cmp_gt_u32 %[lim], %[dl]\n\t"
                "s_cbranch_scc0 5b\n"
                "6:\n\t"
                "ds_read_b128 v[144:147], %[ws]\n\t"
                "ds_read_b128 v[148:151], %[ws] offset:4096\n\t"
                "ds_read_b128 v[128:131], %[xs]\n\t"
                "ds_read_b128 v[132:135], %[xs] offset:1024\n\t"
                "ds_read_b128 v[136:139], %[xs] offset:2048\n\t"
                "ds_read_b128 v[140:143], %[xs] offset:3072\n"
                "1:\n\t"
                "s_waitcnt lgkmcnt(2)\n\t"
                MF("v[0:15]", "v[128:135]", "v[144:151]")
                "ds_read_b128 v[152:155], %[ws] offset:1024\n\t"
                "ds_read_b128 v[156:159], %[ws] offset:5120\n\t"
                "s_add_u32 %[pos], %[pos], 4\n\t"
                "v_mov_b32 v160, %[fr]\n\t"
                "v_mov_b32 v161, %[pos]\n\t"
                "s_waitcnt lgkmcnt(2)\n\t"
                MF("v[64:79]", "v[136:143]", "v[144:151]")
                "ds_read_b128 v[144:147], %[ws] offset:2048\n\t"
                "ds_read_b128 v[148:151], %[ws] offset:6144\n\t"
                "ds_write_b32 v160, v161\n\t"
                "s_add_u32 %[pos], %[pos], 4\n\t"
                "v_mov_b32 v162, %[pos]\n\t"
                "s_waitcnt lgkmcnt(3)\n\t"
                MF("v[16:31]", "v[128:135]", "v[152:159]")
                "ds_read_b64 v[160:161], v162\n\t"
                MF("v[80:95]", "v[136:143]", "v[152:159]")
                "ds_read_b128 v[152:155], %[ws] offset:3072\n\t"
                "ds_read_b128 v[156:159], %[ws] offset:7168\n\t"
                "s_waitcnt lgkmcnt(2)\n\t"
                MF("v[32:47]", "v[128:135]", "v[144:151]")
                "v_readfirstlane_b32 %[lim], v160\n\t"
                "v_readfirstlane_b32 %[lim2], v161\n\t"
                "v_add3_u32 %[xs], v160, %[vl], %[xo]\n\t"
                "v_add3_u32 %[ws], v161, %[vl], %[wo]\n\t"
                "v_mov_b32 v160, %[fr]\n\t"
                "s_min_u32 %[lim], %[lim], %[lim2]\n\t"
                "s_cmp_lt_u32 %[pos], %[end]\n\t"
                "s_cselect_b32 %[lim], %[lim], 0\n\t"
                "s_cmp_lg_u32 %[lim], 0\n\t"
                MF("v[96:111]", "v[136:143]", "v[144:151]")
                "s_cbranch_scc0 2f\n\t"
                "ds_read_b128 v[144:147], %[ws]\n\t"
                "ds_read_b128 v[148:151], %[ws] offset:4096\n\t"
                "s_waitcnt lgkmcnt(2)\n\t"
                MF("v[48:63]", "v[128:135]", "v[152:159]")
                "ds_read_b128 v[128:131], %[xs]\n\t"
                "ds_read_b128 v[132:135], %[xs] offset:1024\n\t"
                MF("v[112:127]", "v[136:143]", "v[152:159]")
                "ds_read_b128 v[136:139], %[xs] offset:2048\n\t"
                "ds_read_b128 v[140:143], %[xs] offset:3072\n\t"
                "s_branch 1b\n"
                "2:\n\t"
                "s_waitcnt lgkmcnt(0)\n\t"
                MF("v[48:63]", "v[128:135]", "v[152:159]")
                "ds_write_b32 v160, v162\n\t"
                MF("v[112:127]", "v[136:143]", "v[152:159]")
                "s_cmp_lg_u32 %[pos], %[end]\n\t"
                "s_cbranch_scc1 5b\n\t"
                "s_cmp_eq_u32 %[end], %[last]\n\t"
                "s_cbranch_scc1 7f\n\t"
                "v_mov_b32 v162, %[jump]\n\t"
                "ds_write_b32 v160, v162\n\t"
                "s_mov_b32 %[pos], %[jump]\n\t"
                "s_mov_b32 %[end], %[last]\n\t"
                "s_branch 5b\n"
                "7:\n\t"
                "v_bfrev_b32 v162, -2\n\t"
                "ds_write_b32 v160, v162\n\t"
                "v_mov_b32 v161, %[ab]\n\t"
                "ds_read_b32 v161, v161\n\t"
                "v_and_b32 v162, 0x200, %[vl]\n\t"
                "v_lshlrev_b32 v162, 7, v162\n\t"
                "v_and_b32 %[xs], 0x1f0, %[vl]\n\t"
                "v_or_b32 %[xs], %[xs], v162\n\t"
                "s_waitcnt lgkmcnt(0)\n\t"
                "v_readfirstlane_b32 %[lim], v161\n\t"
                "s_cmp_lg_u32 %[lim], 0\n\t"
                "s_cselect_b32 %[lim], 0x7fc00000, %[scale]\n\t"
                "s_cmp_eq_u32 %[last], %[la]\n\t"
                "s_cselect_b64 vcc, -1, 0\n\t"
                "s_nop 15\n\t"
                "s_cmp_lt_u32 %[last], %[lc]\n\t"
                ".set gk_i, 0\n\t"
                ".rept 32\n\t"
                ".set gk_a, (gk_i >> 4) * 64 + (gk_i & 15)\n\t"
                ".set gk_t, 128 + (gk_i & 7) * 4\n\t"
                "v_mul_f32 v[gk_t], %[lim], v[gk_a]\n\t"
                "v_mul_f32 v[gk_t + 1], %[lim], v[gk_a + 16]\n\t"
                "v_mul_f32 v[gk_t + 2], %[lim], v[gk_a + 32]\n\t"
                "v_mul_f32 v[gk_t + 3], %[lim], v[gk_a + 48]\n\t"
                "s_mov_b32 %[lim2], ((gk_i >> 4) * 32 + (gk_i & 3) + 8 * ((gk_i >> 2) & 3)) * 16384\n\t"
                "s_cbranch_scc0 6\n\t"
                "buffer_store_dwordx4 v[gk_t:gk_t + 3], %[xs], %[ro], %[lim2] offen " EPIE "\n\t"
                "s_sleep %[epa]\n\t"
                "s_cbranch_vccnz 4\n\t"
                "s_sleep %[eps]\n\t"
                "s_branch 2\n\t"
                "buffer_store_dwordx4 v[gk_t:gk_t + 3], %[xs], %[ro], %[lim2] offen " EPIF "\n\t"
                ".set gk_i, gk_i + 1\n\t"
                ".endr"
                : [xs] "=&v"(xs), [ws] "=&v"(wsl), [pos] "+&s"(pos), [end] "+&s"(end),
                  [spins] "+&s"(spins), [lim] "=&s"(lim), [lim2] "=&s"(lim2)
                : [vl] "v"(vl), [xo] "s"((wm - XB) * UNIT), [wo] "s"(wn * 8192), [jump] "s"(jump), [ab] "n"(L_ABORT), [fr] "s"(freep), [last] "s"(last), [t0] "s"(t0), [dl] "n"(3 * DEADLINE),
                  [lc] "n"(L_D + 8 * NE), [la] "n"(ANS ? L_D + 8 * 64 : 0), [scale] "s"(scale), [ro] "s"(ro), [eps] "n"(EPS - EPA), [epa] "n"(EPA)
                : "memory", "scc", "vcc",
                  "v0", "v1", "v2", "v3", "v4", "v5", "v6", "v7", "v8", "v9", "v10", "v11", "v12", "v13",
                  "v14", "v15", "v16", "v17", "v18", "v19", "v20", "v21", "v22", "v23", "v24", "v25", "v26",
                  "v27", "v28", "v29", "v30", "v31", "v32", "v33", "v34", "v35", "v36", "v37", "v38", "v39",
                  "v40", "v41", "v42", "v43", "v44", "v45", "v46", "v47", "v48", "v49", "v50", "v51", "v52",
                  "v53", "v54", "v55", "v56", "v57", "v58", "v59", "v60", "v61", "v62", "v63", "v64", "v65",
                  "v66", "v67", "v68", "v69", "v70", "v71", "v72", "v73", "v74", "v75", "v76", "v77", "v78",
                  "v79", "v80", "v81", "v82", "v83", "v84", "v85", "v86", "v87", "v88", "v89", "v90", "v91",
                  "v92", "v93", "v94", "v95", "v96", "v97", "v98", "v99", "v100", "v101", "v102", "v103",
                  "v104", "v105", "v106", "v107", "v108", "v109", "v110", "v111", "v112", "v113", "v114",
                  "v115", "v116", "v117", "v118", "v119", "v120", "v121", "v122", "v123", "v124", "v125",
                  "v126", "v127", "v128", "v129", "v130", "v131", "v132", "v133", "v134", "v135", "v136",
                  "v137", "v138", "v139", "v140", "v141", "v142", "v143", "v144", "v145", "v146", "v147",
                  "v148", "v149", "v150", "v151", "v152", "v153", "v154", "v155", "v156", "v157", "v158",
                  "v159", "v160", "v161", "v162");
        }
    }
}

extern "C" void kernel_launch(int rank, int nranks, ncclComm_t comm,
                              hipStream_t stream,
                              void* const* d_in, const int* in_sizes, int n_in,
                              void* d_out, int out_size,
                              const int* in_dtypes, int out_dtype,
                              void* const* d_out_all,
                              void* const* d_ws_all, size_t ws_bytes) {
    if (nranks != NR || (unsigned)rank >= (unsigned)NR || n_in != 4 || ws_bytes < WS_NEED ||
        in_sizes[0] != MN * KS || in_sizes[1] != MN * KS || out_size != MN * MN) {
        fprintf(stderr, "kernel_launch: unexpected problem shape\n");
        return;
    }
    Params P;
    P.x = (const unsigned char*)d_in[0];
    P.w = (const unsigned char*)d_in[1];
    P.sx = (const float*)d_in[2];
    P.sw = (const float*)d_in[3];
    P.out = (float*)d_out;
    for (int p = 0; p < NR; ++p) P.ws[p] = (char*)d_ws_all[(rank + p) & 7];
    P.rank = rank;
    hipLaunchKernelGGL(gemm_ag_kernel, dim3(NWG), dim3(NTHREADS), 0, stream, P);
    HIP_CHECK(hipGetLastError());
}
```
